```python
import numpy as np

kernel_name = "gpu_gemm_a2a_m4096_k4096_n16384_e4m3e4m3_dqsilu"


ABSMAX_RTOL = 0.03

COMPUTE_DTYPE = "float32"
EPILOGUE = "dequant_silu"
IN_QUANT = None
OUT_QUANT = None
IN_FP8 = "e4m3e4m3"
IN_DTYPES = {"x": "e4m3", "w_mat": "e4m3",
             "scale_x": "float32", "scale_w": "float32"}
WORLD_SIZE = 8

def _snap_fp8(x, mant_bits, max_abs, min_normal):
    a = np.abs(x.astype(np.float64))
    _m, _e = np.frexp(a)
    q = np.where(a >= min_normal, np.ldexp(1.0, _e - 1 - mant_bits),
                 min_normal * 2.0 ** -mant_bits)
    snapped = np.minimum(np.round(a / q) * q, max_abs)
    return (np.sign(x) * snapped).astype(np.float32)

def setup_inputs():

    x = _snap_fp8(np.random.randn(4096, 4096).astype(np.float32), 3, 448.0, 2.0 ** -6)
    w_mat = _snap_fp8(np.random.randn(4096, 16384).astype(np.float32), 3, 448.0, 2.0 ** -6)
    scale_x = (0.02 * (1.0 + np.random.rand(1))).astype(np.float32)
    scale_w = (0.02 * (1.0 + np.random.rand(1))).astype(np.float32)
    return {"x": x, "w_mat": w_mat, "scale_x": scale_x, "scale_w": scale_w}

def reference(x, w_mat, scale_x, scale_w):
    acc = (x @ w_mat).astype(np.float32)
    y = (acc * (scale_x * scale_w)).astype(np.float32)
    return (y / (1.0 + np.exp(-np.clip(y, -60.0, 60.0)))).astype(np.float32)
```

```cpp
#include <hip/hip_runtime.h>
#include <rccl/rccl.h>
#include <cstdio>

#define HIP_CHECK(expr) do { hipError_t _e = (expr); if (_e != hipSuccess) { \
    fprintf(stderr, "HIP error %s at %s:%d\n", hipGetErrorString(_e), __FILE__, __LINE__); \
    return; } } while (0)

#define DD 4

namespace {

typedef unsigned u32x2 __attribute__((ext_vector_type(2)));
typedef unsigned u32x4 __attribute__((ext_vector_type(4)));
typedef int v2i __attribute__((ext_vector_type(2)));
typedef int v4i __attribute__((ext_vector_type(4)));
typedef int v8i __attribute__((ext_vector_type(8)));
typedef float v4f __attribute__((ext_vector_type(4)));
#define LAS __attribute__((address_space(3)))

constexpr int NR = 8, KDIM = 4096, NGLOB = 16384, MS = 512, NS = 2048;
constexpr int KT = 32;
constexpr int BM = 128, BN = 128, SROWS = 256;
constexpr int MI = 4, NI = 2;
constexpr int SLOT = 16384, NA = 4, NB = 5;
constexpr int LDS_A = 0, LDS_B = NA * SLOT, LDS_CTL = LDS_B + NB * SLOT;
constexpr int LDS_BYTES = LDS_CTL + 16;
constexpr int GRID = 256, THREADS = 576;

constexpr int UI = 4, U = UI * 1024, UROWS = U / 128;
constexpr int UPS = SROWS / UROWS;
constexpr int UNITS = MS * KDIM / U;
constexpr int NU = BM / UROWS;
constexpr int NPUSH = 28, PG = NPUSH / 7, PUSH_N = UNITS / PG;
static_assert(PUSH_N % DD == 0 && (DD - 1) * (2 * UI + 1) + UI <= 63 && NU == 4 && NA == 4 && NB == 5, "geometry");

constexpr size_t OFF_CNT = 0, OFF_TMO = 256, OFF_DUMMY = 4096, OFF_FLAGS = 65536, OFF_DBG = 512 * 1024, OFF_X = 1 << 20;
constexpr size_t X_SLOT = (size_t)NR * MS * KDIM, WS_NEED = OFF_X + 2 * X_SLOT;
constexpr int FLAG_GROUP = KT * NU * 4;
constexpr unsigned long long DEADLINE_TICKS = 50000000ull;

struct Args {
  const unsigned char* x; const unsigned char* w; const float* sx; const float* sw; float* out;
  char* ws[NR]; int rank; int pad;
};

template <int N> __device__ __forceinline__ void wait_vm() { asm volatile("s_waitcnt vmcnt(%0)" :: "i"(N) : "memory"); }
__device__ __forceinline__ void barrier() { asm volatile("s_barrier" ::: "memory"); }

template <bool SYS> __device__ __forceinline__ void dma1(const void* base, unsigned voff, unsigned lds) {
  if constexpr (SYS)
    asm volatile("s_mov_b32 m0, %2\n\ts_nop 0\n\tglobal_load_lds_dwordx4 %0, %1 sc0 sc1" :: "v"(voff), "s"(base), "s"(lds) : "memory");
  else
    asm volatile("s_mov_b32 m0, %2\n\ts_nop 0\n\tglobal_load_lds_dwordx4 %0, %1" :: "v"(voff), "s"(base), "s"(lds) : "memory");
}
__device__ __forceinline__ char* pick_ws(const Args& a, unsigned r) {
  char* p = a.ws[0];
#pragma unroll
  for (unsigned i = 1; i < NR; ++i) p = (r == i) ? a.ws[i] : p;
  return p;
}
#define STAMP(idx) do { } while (0)
#define LSTAMP(idx) do { } while (0)

struct Push { const unsigned char* x; char* dst; char* flags; unsigned voff[UI]; unsigned lane16; };
__device__ __forceinline__ char* unit_flag(const Push& p, unsigned q) {
  const unsigned sk = q / UPS, piece = q % UPS;
  return p.flags + ((sk / KT) * 2 + piece / NU) * FLAG_GROUP + ((sk % KT) * NU + piece % NU) * 4;
}
__device__ __forceinline__ void unit_load(const Push& p, unsigned q, u32x4 (&buf)[UI]) {
  const unsigned sk = q / UPS, piece = q % UPS;
  const unsigned char* s = p.x + ((sk / KT) * SROWS + piece * UROWS) * KDIM + (sk % KT) * 128;
#pragma unroll
  for (int i = 0; i < UI; ++i) asm volatile("global_load_dwordx4 %0, %1, %2" : "=v"(buf[i]) : "v"(p.voff[i]), "s"(s) : "memory");
}
__device__ __forceinline__ void unit_store(const Push& p, unsigned q, const u32x4 (&buf)[UI]) {
  const char* d = p.dst + (size_t)q * U;
#pragma unroll
  for (int i = 0; i < UI; ++i)
    asm volatile("global_store_dwordx4 %0, %1, %2 offset:%3 sc0 sc1\n\ts_nop 1" :: "v"(p.lane16), "v"(buf[i]), "s"(d), "i"(i * 1024) : "memory");
}
__device__ __forceinline__ void flag_store(char* f, unsigned zero, unsigned val) {
  asm volatile("s_mov_b64 exec, 1\n\tglobal_store_dword %0, %1, %2 sc0 sc1\n\ts_mov_b64 exec, -1" :: "v"(zero), "v"(val), "s"(f) : "memory");
}
template <int T> __device__ __forceinline__ void push_tail(const Push& p, unsigned q0, unsigned zero, unsigned call) {
  if constexpr (T < DD) {
    wait_vm<UI + (DD - 1 - T) * (2 * UI + 1) + T>();
    flag_store(unit_flag(p, q0 + (PUSH_N - DD + T) * PG), zero, call);
    push_tail<T + 1>(p, q0, zero, call);
  }
}
__device__ __forceinline__ unsigned read_call(char* my) {
  unsigned cnt;
  asm volatile("global_load_dword %0, %1, off sc1\n\ts_waitcnt vmcnt(0)" : "=&v"(cnt) : "v"(my + OFF_CNT) : "memory");
  return (unsigned)__builtin_amdgcn_readfirstlane(cnt) / GRID + 1;
}

__global__ __launch_bounds__(THREADS, 3) void gemm_ag_kernel(Args a) {
  extern __shared__ __attribute__((aligned(16))) char smem[];
  const int lane = threadIdx.x & 63, wave = __builtin_amdgcn_readfirstlane(threadIdx.x >> 6), b = blockIdx.x;
  const unsigned lds0 = (unsigned)(size_t)(LAS char*)smem;
  char* const my = pick_ws(a, (unsigned)a.rank);
  const int xcd = b & 7, idx = b >> 3;

  if (idx < 4) {
    if (wave != 0) return;
    const unsigned pid = idx * 8 + xcd;
    if (pid < NPUSH) {
      u32x4 pbuf[DD][UI];
      Push ps;
      const unsigned q0 = pid / 7u;
      ps.x = a.x;
      ps.lane16 = lane * 16;
#pragma unroll
      for (int i = 0; i < UI; ++i) ps.voff[i] = (8 * i + (lane >> 3)) * KDIM + (lane & 7) * 16;
#pragma unroll
      for (int t = 0; t < DD; ++t) unit_load(ps, q0 + t * PG, pbuf[t]);
      const unsigned call = read_call(my);
      char* const peer = pick_ws(a, (unsigned)(a.rank + 1 + pid % 7u) & 7u);
      ps.dst = peer + OFF_X + (size_t)(call & 1u) * X_SLOT + (size_t)a.rank * (MS * KDIM);
      ps.flags = peer + OFF_FLAGS + (size_t)a.rank * 4 * FLAG_GROUP;
      char* const dummy = my + OFF_DUMMY + pid * 128;
      const unsigned zero = 0;
      for (unsigned j0 = 0; j0 < PUSH_N; j0 += DD) {
#pragma unroll
        for (int t = 0; t < DD; ++t) {
          const unsigned q = q0 + (j0 + t) * PG;
          wait_vm<(DD - 1) * (2 * UI + 1)>();
#pragma unroll
          for (int i = 0; i < UI; ++i) asm volatile("" : "+v"(pbuf[t][i]));
          flag_store(j0 > 0 ? unit_flag(ps, q - DD * PG) : dummy, zero, call);
          unit_store(ps, q, pbuf[t]);
          unit_load(ps, j0 + DD < PUSH_N ? q + DD * PG : q, pbuf[t]);
        }
      }
      push_tail<0>(ps, q0, zero, call);
      wait_vm<0>();
#pragma unroll
      for (int t = 0; t < DD; ++t)
#pragma unroll
        for (int i = 0; i < UI; ++i) asm volatile("" :: "v"(pbuf[t][i]));
      STAMP(16);
    }
    if (lane == 0) __hip_atomic_fetch_add((unsigned*)(my + OFF_CNT), 1u, __ATOMIC_RELAXED, __HIP_MEMORY_SCOPE_AGENT);
    return;
  }

  const int si = (idx - 4) >> 2, rsub = xcd >> 2, cblk = (xcd & 3) * 4 + (idx & 3);
  const int src = (a.rank + 1 + si) & 7;
  const int first = si < 2 ? 0 : 1;
  const int total = (3 - first) * KT;

  if (wave == 8) {
    unsigned* const tmo = (unsigned*)(my + OFF_TMO);
    const unsigned call = read_call(my);
    if (lane == 0) {
      __hip_atomic_fetch_add((unsigned*)(my + OFF_CNT), 1u, __ATOMIC_RELAXED, __HIP_MEMORY_SCOPE_AGENT);
      ((LAS unsigned*)(smem + LDS_CTL))[0] = 0;
      ((LAS unsigned*)(smem + LDS_CTL))[1] = call;
    }
    bool dead = false;
    int ready = 0;
    u32x2 v = {0u, 0u};
    for (int t = 0; t < total; ++t) {
      const int job = first + t / KT, kt = t % KT;
      if (t == NA) barrier();
      if (job > 0 && !dead) {
        const char* fl = my + OFF_FLAGS + (src * 4 + (job - 1) * 2 + rsub) * FLAG_GROUP + lane * 8;
        if (kt == 0) ready = 0;
        else if (ready < KT) {
          asm volatile("s_waitcnt vmcnt(0)" : "+v"(v) :: "memory");
          const unsigned long long m = ~__ballot(v[0] >= call && v[1] >= call);
          ready = (m ? __builtin_ctzll(m) : 64) >> 1;
        }
        if (ready <= kt) {
          const unsigned long long t0 = __builtin_amdgcn_s_memrealtime();
          for (unsigned polls = 1;; ++polls) {
            asm volatile("global_load_dwordx2 %0, %1, off sc0 sc1\n\ts_waitcnt vmcnt(0)" : "=&v"(v) : "v"(fl) : "memory");
            const unsigned long long m = ~__ballot(v[0] >= call && v[1] >= call);
            ready = (m ? __builtin_ctzll(m) : 64) >> 1;
            if (ready > kt) break;
            if ((polls & 31u) == 0 &&
                (__hip_atomic_load(tmo, __ATOMIC_RELAXED, __HIP_MEMORY_SCOPE_AGENT) == call
                 || __builtin_amdgcn_s_memrealtime() - t0 > DEADLINE_TICKS)) {
              __hip_atomic_store(tmo, call, __ATOMIC_RELAXED, __HIP_MEMORY_SCOPE_AGENT);
              dead = true;
              break;
            }
          }
          LSTAMP(t);
        }
        if (ready < KT && kt < KT - 1 && !dead)
          asm volatile("global_load_dwordx2 %0, %1, off sc0 sc1" : "=&v"(v) : "v"(fl) : "memory");
        if (kt == KT - 1 && !dead) dead = __hip_atomic_load(tmo, __ATOMIC_RELAXED, __HIP_MEMORY_SCOPE_AGENT) == call;
        if (dead && lane == 0) *(LAS unsigned*)(smem + LDS_CTL) = 1;
      }
      asm volatile("s_waitcnt lgkmcnt(0)\n\ts_barrier" ::: "memory");
    }
#pragma unroll
    for (int t = 0; t < NA; ++t) barrier();
    return;
  }

  const int wm = wave >> 2, wn = wave & 3, g = lane >> 4, l16 = lane & 15;
  const int xoff = (wm * 64 + l16) * 128 + ((g ^ ((l16 >> 1) & 7)) << 4);
  const int sB = ((l16 >> 2) & 3) | ((g & 1) << 2);
  int woff[NI];
#pragma unroll
  for (int ni = 0; ni < NI; ++ni) woff[ni] = (16 * g + (l16 >> 1)) * 128 + 8 * (l16 & 1) + (((wn * NI + ni) ^ sB) << 4);

  const unsigned achunk0 = ((lane & 7) ^ (lane >> 4)) << 4, achunk1 = achunk0 ^ 64u;
  const unsigned wv = (lane >> 3) * NGLOB + (((lane & 7) ^ (((wave & 1) << 2) | (lane >> 4))) << 4);
  const unsigned char* const wbase = a.w + (size_t)(16 * wave) * NGLOB + a.rank * NS + cblk * BN;
  const unsigned char* xremote;
  const unsigned char* aptr; unsigned av0, av1, a8, aks;
  auto set_job = [&](int job) {
    if (job == 0) {
      aptr = a.x + (size_t)((2 * rsub + si) * BM + 16 * wave) * KDIM;
      av0 = (lane >> 3) * KDIM + achunk0; av1 = (lane >> 3) * KDIM + achunk1; a8 = 8 * KDIM; aks = 128;
    } else {
      aptr = xremote + (size_t)(job - 1) * (SROWS * KDIM);
      av0 = (lane >> 3) * 128 + achunk0; av1 = (lane >> 3) * 128 + achunk1; a8 = 8 * 128; aks = SROWS * 128;
    }
  };
  int ib = 0, sa_w = 0, sb_w = 0, sa_r = 0, sb_r = 0;
  auto issue_a = [&](int half) {
    const unsigned dst = lds0 + LDS_A + sa_w * SLOT + wave * 2048;
    if (half == 0) dma1<true>(aptr, av0, dst);
    else { dma1<true>(aptr + a8, av1, dst + 1024); sa_w = (sa_w + 1) & (NA - 1); aptr += aks; }
  };
  auto issue_b = [&](int half) {
    const unsigned char* p = wbase + (size_t)(ib & (KT - 1)) * (128 * NGLOB);
    const unsigned dst = lds0 + LDS_B + sb_w * SLOT + wave * 2048;
    if (half == 0) dma1<false>(p, wv, dst);
    else { dma1<false>(p + 8 * NGLOB, wv, dst + 1024); sb_w = (sb_w == NB - 1) ? 0 : sb_w + 1; ++ib; }
  };
  v8i xf[2][MI], wf[2][NI];
  auto read_w = [&](v8i& wd, int ni) {
    LAS char* pb = (LAS char*)smem + LDS_B + sb_r * SLOT;
#pragma unroll
    for (int r = 0; r < 4; ++r) {
      const v2i t = __builtin_amdgcn_ds_read_tr8_b64_v2i32((LAS v2i*)(pb + woff[ni] + (r >> 1) * 8192 + (r & 1) * 1024));
      wd[2 * r] = t[0]; wd[2 * r + 1] = t[1];
    }
  };
  auto read_x = [&](v8i& xd, int mi) {
    LAS char* pa = (LAS char*)smem + LDS_A + sa_r * SLOT;
    const v4i lo = *(LAS v4i*)(pa + xoff + mi * 2048), hi = *(LAS v4i*)(pa + (xoff ^ 64) + mi * 2048);
    xd = __builtin_shufflevector(lo, hi, 0, 1, 2, 3, 4, 5, 6, 7);
  };
  auto next_read_slot = [&]() {
    sa_r = (sa_r + 1) & (NA - 1);
    sb_r = (sb_r == NB - 1) ? 0 : sb_r + 1;
  };
  v4f acc[MI][NI];
#define SB __builtin_amdgcn_sched_barrier(0)
#define MMA(H, mi, ni) do { SB; acc[mi][ni] = __builtin_amdgcn_mfma_scale_f32_16x16x128_f8f6f4(wf[H][ni], xf[H][mi], acc[mi][ni], 0, 0, 0, 0, 0, 0); SB; } while (0)
#define SCHED 4
#define STEP_BODY(H, DO_B, DO_A, DO_READ)                                                   \
    MMA(H, 0, 0); if (DO_READ) read_w(wf[(H) ^ 1][0], 0);                                   \
    SB; if (DO_B) issue_b(0);                                                               \
    MMA(H, 0, 1); if (DO_READ) read_w(wf[(H) ^ 1][1], 1);                                   \
    MMA(H, 1, 0); if (DO_READ) read_x(xf[(H) ^ 1][0], 0);                                   \
    SB; if (DO_B) issue_b(1);                                                               \
    MMA(H, 1, 1); if (DO_READ) read_x(xf[(H) ^ 1][1], 1);                                   \
    MMA(H, 2, 0); if (DO_READ) read_x(xf[(H) ^ 1][2], 2);                                   \
    SB; if (DO_A) issue_a(0);                                                               \
    MMA(H, 2, 1); if (DO_READ) read_x(xf[(H) ^ 1][3], 3);                                   \
    MMA(H, 3, 0); if (DO_A) issue_a(1);                                                     \
    MMA(H, 3, 1);
#define STEP(H, DO_B, DO_A, DO_READ, VM) do {                                               \
    barrier();                                                                              \
    STEP_BODY(H, DO_B, DO_A, DO_READ)                                                       \
    if (DO_READ) next_read_slot();                                                          \
    SB;                                                                                     \
    __builtin_amdgcn_s_waitcnt(0xC07F);                                                     \
    if ((VM) >= 0) wait_vm<((VM) >= 0 ? (VM) : 0)>();                                       \
  } while (0)

  issue_b(0); issue_b(1);
  float sx, sw;
  asm volatile("s_load_dword %0, %2, 0x0\n\ts_load_dword %1, %3, 0x0\n\ts_waitcnt lgkmcnt(0)" : "=&s"(sx), "=&s"(sw) : "s"(a.sx), "s"(a.sw) : "memory");
  const float scale = sx * sw;
  barrier();
  const unsigned call = __builtin_amdgcn_readfirstlane(((volatile LAS unsigned*)(smem + LDS_CTL))[1]);
  xremote = (const unsigned char*)my + OFF_X + (size_t)(call & 1u) * X_SLOT + (size_t)src * (MS * KDIM) + (rsub * BM + 16 * wave) * 128;
  set_job(first);
  issue_b(0); issue_b(1); issue_a(0); issue_a(1);
#pragma unroll
  for (int p = 1; p < NA; ++p) { barrier(); issue_b(0); issue_b(1); issue_a(0); issue_a(1); }
  wait_vm<12>();
  barrier();
#pragma unroll
  for (int ni = 0; ni < NI; ++ni) read_w(wf[0][ni], ni);
#pragma unroll
  for (int mi = 0; mi < MI; ++mi) read_x(xf[0][mi], mi);
  next_read_slot();
  __builtin_amdgcn_s_waitcnt(0xC07F);
  wait_vm<8>();

  for (int job = first; job < 3; ++job) {
#pragma unroll
    for (int mi = 0; mi < MI; ++mi)
#pragma unroll
      for (int ni = 0; ni < NI; ++ni) acc[mi][ni] = (v4f){0.f, 0.f, 0.f, 0.f};
    for (int seg = 0; seg < 2; ++seg) {
      const int n = job == 2 ? (seg == 0 ? KT - 6 : 0) : (seg == 0 ? KT - NA : NA);
      for (int kt = 0; kt < n; kt += 2) {
        STEP(0, true, true, true, 8);
        STEP(1, true, true, true, 8);
      }
      if (seg == 0 && job < 2) set_job(job + 1);
    }
    if (job == 2) {
      STEP(0, true, true, true, 8);
      STEP(1, false, true, true, 6);
      STEP(0, false, false, true, 2);
      STEP(1, false, false, true, 0);
      STEP(0, false, false, true, -1);
      STEP(1, false, false, false, -1);
    }
    if (wave == 0) STAMP(2 + 2 * job);
    const float oscale = *(volatile LAS unsigned*)(smem + LDS_CTL) != 0 ? __builtin_nanf("") : scale;
    const float nl2e = -1.44269504088896341f * scale, lim = 60.f * 1.44269504088896341f;
    const int row0 = job == 0 ? a.rank * MS + (2 * rsub + si) * BM : src * MS + (job - 1) * SROWS + rsub * BM;
    float* orow = a.out + (size_t)(row0 + wm * 64 + l16) * NS + cblk * BN + wn * 32 + 4 * g;
#pragma unroll
    for (int mi = 0; mi < MI; ++mi)
#pragma unroll
      for (int ni = 0; ni < NI; ++ni) {
        v4f o;
#pragma unroll
        for (int j = 0; j < 4; ++j) {
          const float v = acc[mi][ni][j];
          const float e = __builtin_amdgcn_exp2f(__builtin_amdgcn_fmed3f(v * nl2e, -lim, lim));
          o[j] = v * oscale * __builtin_amdgcn_rcpf(1.f + e);
        }
        *(v4f*)(orow + (size_t)mi * 16 * NS + ni * 16) = o;
      }
    if (wave == 0) STAMP(3 + 2 * job);
  }
}

}

extern "C" void kernel_launch(int rank, int nranks, ncclComm_t comm, hipStream_t stream,
                              void* const* d_in, const int* in_sizes, int n_in,
                              void* d_out, int out_size, const int* in_dtypes, int out_dtype,
                              void* const* d_out_all, void* const* d_ws_all, size_t ws_bytes) {
  if (nranks != NR || (unsigned)rank >= (unsigned)NR || n_in != 4 || in_sizes[0] != MS * KDIM || in_sizes[1] != KDIM * NGLOB ||
      out_size != NR * MS * NS || ws_bytes < WS_NEED) {
    fprintf(stderr, "unexpected problem geometry\n");
    return;
  }
  HIP_CHECK(hipFuncSetAttribute((const void*)gemm_ag_kernel, hipFuncAttributeMaxDynamicSharedMemorySize, LDS_BYTES));
  Args a;
  a.x = (const unsigned char*)d_in[0]; a.w = (const unsigned char*)d_in[1];
  a.sx = (const float*)d_in[2]; a.sw = (const float*)d_in[3]; a.out = (float*)d_out;
  for (int i = 0; i < NR; ++i) a.ws[i] = (char*)d_ws_all[i];
  a.rank = rank; a.pad = 0;
  gemm_ag_kernel<<<GRID, THREADS, LDS_BYTES, stream>>>(a);
  HIP_CHECK(hipGetLastError());
}
```

```cpp
#include <hip/hip_runtime.h>
#include <rccl/rccl.h>
#include <cstdio>

#define HIP_CHECK(expr) do { hipError_t _e = (expr); if (_e != hipSuccess) { \
    fprintf(stderr, "HIP error %s at %s:%d\n", hipGetErrorString(_e), __FILE__, __LINE__); \
    return; } } while (0)

#define DD 4
#define GS 2
#define DK 4
#define DLOOK 2

namespace {

typedef int v2i __attribute__((ext_vector_type(2)));
typedef int v4i __attribute__((ext_vector_type(4)));
typedef int v8i __attribute__((ext_vector_type(8)));
typedef float v2f __attribute__((ext_vector_type(2)));
typedef float v4f __attribute__((ext_vector_type(4)));
#define LAS __attribute__((address_space(3)))

constexpr int NR = 8, KDIM = 4096, NGLOB = 16384, MS = 512, NS = 2048;
constexpr int KT = 32;
constexpr int BM = 128, BN = 128, SROWS = 256;
constexpr int MI = 4, NI = 2;
constexpr int SLOT = 16384, NA = 4, NB = 5;
constexpr int NE = 3, ESLOT = 4096;
constexpr int LDS_A = 0, LDS_B = NA * SLOT, LDS_CTL = LDS_B + NB * SLOT + NE * ESLOT;
constexpr int LDS_POLL = LDS_CTL + 16, LDS_TMO = LDS_POLL + 512, LDS_JUNK = LDS_TMO + 512, LDS_BYTES = LDS_JUNK + 256;
constexpr int GRID = 256, THREADS = 768;
constexpr int PW = 4;
constexpr int SKEW = 1;
constexpr int LEAD = 2;
static_assert(DD + DLOOK + DK + 1 <= 16 && LDS_BYTES <= 163840, "queue and LDS size");

constexpr size_t OFF_CNT = 0, OFF_TMO = 256, OFF_TICKET = 1024, OFF_FLAGS = 65536, OFF_X = 1 << 20;
constexpr size_t X_SLOT = (size_t)NR * MS * KDIM, WS_NEED = OFF_X + 2 * X_SLOT;
constexpr int FLAG_SRC = 4096;
constexpr unsigned long long DEADLINE_TICKS = 50000000ull;

struct Args {
  const unsigned char* x; const unsigned char* w; const float* sx; const float* sw; float* out;
  char* my; char* peer[NR - 1];
  int rank; int pad;
  unsigned first[PW];
  unsigned order[64];
};

template <int N> __device__ __forceinline__ void wait_vm() { asm volatile("s_waitcnt vmcnt(%0)" :: "i"(N) : "memory"); }
__device__ __forceinline__ void barrier() { asm volatile("s_barrier" ::: "memory"); }

__device__ __forceinline__ void dma_half(const unsigned char* p, unsigned v0, unsigned v1, unsigned lds) {
#define DMA4(b) "global_load_lds_dwordx4 %0, " b " sc0 sc1\n\tglobal_load_lds_dwordx4 %1, " b " offset:1024 sc0 sc1\n\t" \
                "global_load_lds_dwordx4 %0, " b " offset:2048 sc0 sc1\n\tglobal_load_lds_dwordx4 %1, " b " offset:3072 sc0 sc1\n\t"
  asm volatile("s_mov_b32 m0, %4\n\ts_nop 0\n\t" DMA4("%2") "s_add_u32 m0, m0, 0x1000\n\ts_nop 0\n\t" DMA4("%3")
               :: "v"(v0), "v"(v1), "s"(p), "s"(p + 4096), "s"(lds) : "memory", "scc");
}
__device__ __forceinline__ void wait_vm_n(unsigned n) {
  n = __builtin_amdgcn_readfirstlane(n);
  asm volatile("s_min_u32 %0, %0, 47\n\ts_lshl_b32 %0, %0, 3\n\ts_add_u32 %0, %0, 12\n\ts_getpc_b64 s[40:41]\n\ts_add_u32 s40, s40, %0\n\t"
               "s_addc_u32 s41, s41, 0\n\ts_setpc_b64 s[40:41]\n\t"
               ".set WVI, 0\n\t.rept 48\n\ts_waitcnt vmcnt(WVI)\n\ts_branch 1f\n\t.set WVI, WVI+1\n\t.endr\n1:"
               : "+s"(n) :: "memory", "scc", "s40", "s41");
}
__device__ __forceinline__ void touch(const void* base, unsigned voff, unsigned lds) {
  asm volatile("s_mov_b32 m0, %2\n\ts_nop 0\n\tglobal_load_lds_dword %0, %1" :: "v"(voff), "s"(base), "s"(lds) : "memory");
}

#define STR2(x) #x
#define STR(x) STR2(x)
__device__ __forceinline__ void push_stream(const unsigned char* x, char* dst, char* flags, char* cnt, unsigned g0, unsigned g1, int lane) {
  const unsigned vo = (lane >> 3) * 4096 + (lane & 7) * 16;
  asm volatile(
    ".set PD, " STR(DD) "\n"
    ".set PG, " STR(GS) "\n"
    ".set PN, PD/PG\n"
    ".macro PLOAD t\n"
    "  s_min_u32 s40, s[60+\\t], 511\n"
    "  s_lshr_b32 s41, s40, 8\n"
    "  s_lshl_b32 s41, s41, 20\n"
    "  s_and_b32 s42, s40, 7\n"
    "  s_lshl_b32 s42, s42, 17\n"
    "  s_bfe_u32 s40, s40, 0x50003\n"
    "  s_lshl_b32 s40, s40, 7\n"
    "  s_add_u32 s40, s40, s41\n"
    "  s_add_u32 s40, s40, s42\n"
    "  s_add_u32 s42, s44, s40\n"
    "  s_addc_u32 s43, s45, 0\n"
    "  global_load_dwordx4 v[2+16*\\t:5+16*\\t], %[v0], s[42:43]\n"
    "  global_load_dwordx4 v[6+16*\\t:9+16*\\t], %[v1], s[42:43]\n"
    "  global_load_dwordx4 v[10+16*\\t:13+16*\\t], %[v2], s[42:43]\n"
    "  global_load_dwordx4 v[14+16*\\t:17+16*\\t], %[v3], s[42:43]\n"
    ".endm\n"
    ".macro PITER t, first\n"
    "  .if \\first\n"
    "  s_waitcnt vmcnt(4*(PD-1-\\t)+8*\\t+\\t/PG)\n"
    "  .endif\n"
    "  .if (\\t & (PG-1)) == 0\n"
    "  s_cmp_ge_u32 s[60+\\t], 512\n"
    "  s_cbranch_scc1 2f\n"
    "  ds_read_b32 v114, v115\n"
    "  .endif\n"
    "  .if (\\t & (PG-1)) == PG-1\n"
    "  s_waitcnt vmcnt(8*(PD-1)+PN-1)\n"
    "  v_mov_b32 v110, s70\n"
    "  s_mov_b64 exec, 1\n"
    "  global_store_dword v110, v111, %[fl] sc0 sc1\n"
    "  s_mov_b64 exec, -1\n"
    "  .set PI, 0\n"
    "  .rept PN-1\n"
    "  s_mov_b32 s[70+PI], s[71+PI]\n"
    "  .set PI, PI+1\n"
    "  .endr\n"
    "  s_lshr_b32 s40, s[60+\\t], 3\n"
    "  s_lshl_b32 s40, s40, 2\n"
    "  s_and_b32 s41, s[60+\\t], 4\n"
    "  s_lshl_b32 s41, s41, 8\n"
    "  s_add_u32 s40, s40, s41\n"
    "  s_and_b32 s41, s[60+\\t], 4-PG\n"
    "  s_mul_i32 s41, s41, 256/PG\n"
    "  s_add_u32 s[70+PN-1], s40, s41\n"
    "  .else\n"
    "  s_waitcnt vmcnt(8*(PD-1)+PN)\n"
    "  .endif\n"
    "  s_lshl_b32 s40, s[60+\\t], 12\n"
    "  s_add_u32 s42, s46, s40\n"
    "  s_addc_u32 s43, s47, 0\n"
    "  global_store_dwordx4 %[l16], v[2+16*\\t:5+16*\\t], s[42:43] sc0 sc1\n"
    "  global_store_dwordx4 %[l16], v[6+16*\\t:9+16*\\t], s[42:43] offset:1024 sc0 sc1\n"
    "  global_store_dwordx4 %[l16], v[10+16*\\t:13+16*\\t], s[42:43] offset:2048 sc0 sc1\n"
    "  global_store_dwordx4 %[l16], v[14+16*\\t:17+16*\\t], s[42:43] offset:3072 sc0 sc1\n"
    "  .if (\\t & (PG-1)) == 0\n"
    "  s_add_u32 s48, s48, 1\n"
    "3:\n"
    "  s_waitcnt lgkmcnt(0)\n"
    "  v_readfirstlane_b32 s49, v114\n"
    "  s_nop 3\n"
    "  s_lshr_b32 s40, s49, 16\n"
    "  s_cmp_eq_u32 s40, s48\n"
    "  s_cbranch_scc1 4f\n"
    "  ds_read_b32 v114, v115\n"
    "  s_branch 3b\n"
    "4:\n"
    "  s_and_b32 s49, s49, 0xffff\n"
    "  s_mul_i32 s49, s49, PG\n"
    "  v_mov_b32 v116, s48\n"
    "  ds_write_b32 v117, v116\n"
    "  s_and_b32 s40, s48, 15\n"
    "  s_lshl_b32 s40, s40, 2\n"
    "  v_mov_b32 v115, s40\n"
    "  .endif\n"
    "  s_add_u32 s[60+\\t], s49, \\t & (PG-1)\n"
    "  PLOAD \\t\n"
    ".endm\n"
    "  s_mov_b64 s[44:45], %[x]\n"
    "  v_mov_b32 v112, 0\n"
    "  global_load_dword v111, v112, %[cnt] sc1\n"
    "  .set PI, 0\n"
    "  .rept PD\n"
    "  .if PI < PG\n"
    "  s_mul_i32 s40, %[g0], PG\n"
    "  .else\n"
    "  s_mul_i32 s40, %[g1], PG\n"
    "  .endif\n"
    "  s_add_u32 s[60+PI], s40, PI & (PG-1)\n"
    "  s_mov_b32 s[70+PI], 2048\n"
    "  PLOAD PI\n"
    "  .set PI, PI+1\n"
    "  .endr\n"
    "  s_mov_b32 s48, 0\n"
    "  v_mov_b32 v115, 0\n"
    "  v_mov_b32 v117, 64\n"
    "  s_waitcnt lgkmcnt(0)\n"
    "  s_barrier\n"
    "  s_waitcnt vmcnt(4*PD)\n"
    "  v_readfirstlane_b32 s40, v111\n"
    "  s_nop 3\n"
    "  s_lshr_b32 s40, s40, 8\n"
    "  s_add_u32 s40, s40, 1\n"
    "  v_mov_b32 v111, s40\n"
    "  s_and_b32 s40, s40, 1\n"
    "  s_lshl_b32 s40, s40, 24\n"
    "  s_mov_b64 s[46:47], %[dst]\n"
    "  s_add_u32 s46, s46, s40\n"
    "  s_addc_u32 s47, s47, 0\n"
    "  .set PJ, 0\n"
    "  .rept PD\n"
    "  PITER PJ, 1\n"
    "  .set PJ, PJ+1\n"
    "  .endr\n"
    "1:\n"
    "  .set PJ, 0\n"
    "  .rept PD\n"
    "  PITER PJ, 0\n"
    "  .set PJ, PJ+1\n"
    "  .endr\n"
    "  s_branch 1b\n"
    "2:\n"
    "  .set PI, 0\n"
    "  .rept PN\n"
    "  s_waitcnt vmcnt(4+(PN-1-PI)*(8*PG+1)+PI)\n"
    "  v_mov_b32 v110, s[70+PI]\n"
    "  s_mov_b64 exec, 1\n"
    "  global_store_dword v110, v111, %[fl] sc0 sc1\n"
    "  s_mov_b64 exec, -1\n"
    "  .set PI, PI+1\n"
    "  .endr\n"
    "  s_waitcnt vmcnt(0) lgkmcnt(0)\n"
    ".purgem PLOAD\n"
    ".purgem PITER\n"
    :
    : [x] "s"(x), [dst] "s"(dst), [fl] "s"(flags), [cnt] "s"(cnt), [g0] "s"(g0), [g1] "s"(g1),
      [v0] "v"(vo), [v1] "v"(vo + 8 * 4096), [v2] "v"(vo + 16 * 4096), [v3] "v"(vo + 24 * 4096), [l16] "v"(lane * 16)
    : "memory", "scc", "vcc",
      "s40", "s41", "s42", "s43", "s44", "s45", "s46", "s47", "s48", "s49", "s60", "s61", "s62", "s63", "s70", "s71", "s72", "s73",
      "v2", "v3", "v4", "v5", "v6", "v7", "v8", "v9", "v10", "v11", "v12", "v13", "v14", "v15", "v16", "v17", "v18", "v19",
      "v20", "v21", "v22", "v23", "v24", "v25", "v26", "v27", "v28", "v29", "v30", "v31", "v32", "v33", "v34", "v35", "v36",
      "v37", "v38", "v39", "v40", "v41", "v42", "v43", "v44", "v45", "v46", "v47", "v48", "v49", "v50", "v51", "v52", "v53",
      "v54", "v55", "v56", "v57", "v58", "v59", "v60", "v61", "v62", "v63", "v64", "v65", "v110", "v111", "v112", "v113",
      "v114", "v115", "v116", "v117");
}
__device__ __forceinline__ void deal_units(char* ticket, unsigned first) {
  asm volatile(
    ".set PD, " STR(DD) "\n"
    ".set PK, " STR(DK) "\n"
    ".set PG, " STR(GS) "\n"
    ".set PN, PD/PG\n"
    ".macro DPUT val\n"
    "  s_and_b32 s40, s48, 15\n"
    "  s_lshl_b32 s40, s40, 2\n"
    "  s_add_u32 s48, s48, 1\n"
    "  s_lshl_b32 s41, s48, 16\n"
    "  s_or_b32 s41, s41, \\val\n"
    "  v_mov_b32 v114, s40\n"
    "  v_mov_b32 v115, s41\n"
    "  ds_write_b32 v114, v115\n"
    ".endm\n"
    ".macro DMAP\n"
    "  v_mov_b32 v114, s49\n"
    "  ds_read_u8 v116, v114 offset:256\n"
    "  s_waitcnt lgkmcnt(0)\n"
    "  v_readfirstlane_b32 s49, v116\n"
    "  s_nop 3\n"
    "  DPUT s49\n"
    ".endm\n"
    ".macro DITER t\n"
    "  s_waitcnt vmcnt(PK-1)\n"
    "  v_readfirstlane_b32 s49, v[100+\\t]\n"
    "  s_nop 3\n"
    "  s_add_u32 s49, s49, %[first]\n"
    "  s_cmp_ge_u32 s49, 512/PG\n"
    "  s_cbranch_scc1 2f\n"
    "  DMAP\n"
    "3:\n"
    "  ds_read_b32 v116, v117\n"
    "  s_waitcnt lgkmcnt(0)\n"
    "  v_readfirstlane_b32 s40, v116\n"
    "  s_nop 3\n"
    "  s_sub_u32 s40, s48, s40\n"
    "  s_cmp_le_u32 s40, " STR(DLOOK) "\n"
    "  s_cbranch_scc1 4f\n"
    "  s_sleep 1\n"
    "  s_branch 3b\n"
    "4:\n"
    "  global_atomic_add v[100+\\t], v112, v113, %[tk] sc0\n"
    ".endm\n"
    "  s_mov_b64 exec, 1\n"
    "  v_mov_b32 v112, 0\n"
    "  v_mov_b32 v113, 1\n"
    "  v_mov_b32 v117, 64\n"
    "  s_mov_b32 s48, PN\n"
    "  .set PI, 0\n"
    "  .rept PK\n"
    "  global_atomic_add v[100+PI], v112, v113, %[tk] sc0\n"
    "  .set PI, PI+1\n"
    "  .endr\n"
    "1:\n"
    "  .set PJ, 0\n"
    "  .rept PK\n"
    "  DITER PJ\n"
    "  .set PJ, PJ+1\n"
    "  .endr\n"
    "  s_branch 1b\n"
    "2:\n"
    "  s_waitcnt vmcnt(0)\n"
    "  .set PI, 0\n"
    "  .rept PK\n"
    "  v_readfirstlane_b32 s49, v[100+PI]\n"
    "  s_nop 3\n"
    "  s_add_u32 s49, s49, %[first]\n"
    "  s_cmp_ge_u32 s49, 512/PG\n"
    "  s_cbranch_scc1 5f\n"
    "  DMAP\n"
    "5:\n"
    "  .set PI, PI+1\n"
    "  .endr\n"
    "  .rept PN\n"
    "  DPUT 0xffff\n"
    "  .endr\n"
    "  s_waitcnt vmcnt(0) lgkmcnt(0)\n"
    "  s_mov_b64 exec, -1\n"
    ".purgem DPUT\n"
    ".purgem DMAP\n"
    ".purgem DITER\n"
    :
    : [tk] "s"(ticket), [first] "s"(first)
    : "memory", "scc", "vcc", "s40", "s41", "s48", "s49",
      "v100", "v101", "v102", "v103", "v104", "v105", "v106", "v107", "v112", "v113", "v114", "v115", "v116", "v117");
}
__device__ __forceinline__ unsigned read_call(char* my) {
  unsigned cnt;
  asm volatile("global_load_dword %0, %1, off sc1\n\ts_waitcnt vmcnt(0)" : "=&v"(cnt) : "v"(my + OFF_CNT) : "memory");
  return (unsigned)__builtin_amdgcn_readfirstlane(cnt) / GRID + 1;
}

__device__ __forceinline__ void warm(const Args& a, unsigned lds0, int idx, int xcd, int wave, int lane) {
  const int i = (idx - 4) * 8 + xcd + wave * 224;
  if (i < 256) touch(a.x + (size_t)i * 2 * KDIM, lane * 128, lds0 + LDS_JUNK);
  __builtin_amdgcn_s_sleep(48);
  const int j = (idx - 4) * 8 + wave;
  const unsigned char* wcol = a.w + a.rank * NS + (xcd & 3) * 512;
  if ((j & 1) == (xcd >> 2)) {
    touch(wcol + (size_t)j * 16 * NGLOB, (lane >> 2) * NGLOB + (lane & 3) * 128, lds0 + LDS_JUNK);
    if (j < 32) touch(wcol + (size_t)(224 + j) * 16 * NGLOB, (lane >> 2) * NGLOB + (lane & 3) * 128, lds0 + LDS_JUNK);
  }
}

template <int NX>
__device__ __forceinline__ void compute(const Args& a, char* smem, int wave, int lane, int si, int rsub, int cblk, int src,
                                        int idx, int xcd) {
  const unsigned lds0 = (unsigned)(size_t)(LAS char*)smem;
  const int l16 = lane & 15;
  const int wm = wave >> 2, wn = wave & 3, g = lane >> 4;
  const int xoff = (wm * 64 + l16) * 128 + ((g ^ ((l16 >> 1) & 7)) << 4);
  const int sB = ((l16 >> 2) & 3) | ((g & 1) << 2);
  int woff[NI];
#pragma unroll
  for (int ni = 0; ni < NI; ++ni) woff[ni] = (16 * g + (l16 >> 1)) * 128 + 8 * (l16 & 1) + (((wn * NI + ni) ^ sB) << 4);

  constexpr int NBX = NB + 1 - NX, DL = SKEW + 1 - NX;
  const int erow0 = (2 * si + rsub) * 32, erow1 = 448 + si * 32;
  int sa_r = 0, sb_r = 0, se_r = 0;
  v8i xf[2][MI], wf[2][NI], xe[2];
  auto read_e = [&](int e) {
    LAS char* pe = (LAS char*)smem + LDS_B + NBX * SLOT + se_r * (NX * ESLOT) + e * ESLOT - wm * 48 * 128;
    const v4i lo = *(LAS v4i*)(pe + xoff), hi = *(LAS v4i*)(pe + (xoff ^ 64));
    xe[e] = __builtin_shufflevector(lo, hi, 0, 1, 2, 3, 4, 5, 6, 7);
    if (e == NX - 1) se_r = (se_r == NE - 1) ? 0 : se_r + 1;
  };
  auto read_w = [&](v8i& wd, int ni) {
    LAS char* pb = (LAS char*)smem + LDS_B + sb_r * SLOT;
#pragma unroll
    for (int r = 0; r < 4; ++r) {
      const v2i t = __builtin_amdgcn_ds_read_tr8_b64_v2i32((LAS v2i*)(pb + woff[ni] + (r >> 1) * 8192 + (r & 1) * 1024));
      wd[2 * r] = t[0]; wd[2 * r + 1] = t[1];
    }
  };
  auto read_x = [&](v8i& xd, int mi) {
    LAS char* pa = (LAS char*)smem + LDS_A + sa_r * SLOT;
    const v4i lo = *(LAS v4i*)(pa + xoff + mi * 2048), hi = *(LAS v4i*)(pa + (xoff ^ 64) + mi * 2048);
    xd = __builtin_shufflevector(lo, hi, 0, 1, 2, 3, 4, 5, 6, 7);
  };
  auto next_read_slot = [&]() {
    sa_r = (sa_r + 1) & (NA - 1);
    sb_r = (sb_r == NBX - 1) ? 0 : sb_r + 1;
    asm volatile("" : "+s"(sa_r), "+s"(sb_r), "+s"(se_r));
  };
  v4f c0[MI + 2][NI], c1[MI][NI];
  float sinv, oinv, nl2e, lim;
  auto uni = [](float v) { return __builtin_bit_cast(float, __builtin_amdgcn_readfirstlane(__builtin_bit_cast(int, v))); };
  const int scr = LDS_B + NBX * SLOT + (wn >> 1) * (NX == 2 ? ESLOT : 2 * ESLOT) + wm * 2048 + (wn & 1) * 1024;
  const int sw = scr + (l16 & 7) * 128 + ((((l16 >> 3) * 4 + g) ^ (l16 & 7)) << 4);
  const int sr = scr + (lane >> 3) * 128 + (((lane & 7) ^ (lane >> 3)) << 4);
  const unsigned ov = ((lane >> 3) * NS + cblk * BN + wn * 32) * 4 + (lane & 7) * 16;
  auto split = [&](v4f (&c)[NI]) {
#pragma unroll
    for (int j = 0; j < 4; ++j) {
      const int lo = __float_as_int(c[0][j]), hi = __float_as_int(c[1][j]);
      c[0][j] = __int_as_float(__builtin_amdgcn_update_dpp(lo, hi, 0x128, 0xf, 0xc, false));
      c[1][j] = __int_as_float(__builtin_amdgcn_update_dpp(hi, lo, 0x128, 0xf, 0x3, false));
    }
  };
  auto turn = [&](const v4f& c) {
    *(volatile LAS v4f*)((LAS char*)smem + sw) = c;
    return *(volatile LAS v4f*)((LAS char*)smem + sr);
  };
  v4f t0, t1;
  auto silu = [&](v4f v) {
    v4f o;
#pragma unroll
    for (int j = 0; j < 4; j += 2) {
      const v2f x = {v[j], v[j + 1]}, t = x * nl2e;
      const v2f e = {__builtin_amdgcn_exp2f(__builtin_amdgcn_fmed3f(t[0], -lim, lim)), __builtin_amdgcn_exp2f(__builtin_amdgcn_fmed3f(t[1], -lim, lim))};
      const v2f d = __builtin_elementwise_fma(e, (v2f){oinv, oinv}, (v2f){oinv, oinv});
      const v2f y = x * (v2f){__builtin_amdgcn_rcpf(d[0]), __builtin_amdgcn_rcpf(d[1])};
      o[j] = y[0]; o[j + 1] = y[1];
    }
    return o;
  };
  auto out_rows = [&](int stage, int mi) {
    const int row = mi < MI ? src * MS + stage * SROWS + rsub * BM + wm * 64 + mi * 16 : a.rank * MS + (mi > MI ? erow1 : erow0) + wm * 16;
    return a.out + (size_t)row * NS;
  };
#define PUT(v, mi, half) \
    asm volatile("global_store_dwordx4 %0, %1, %2 sc0 sc1\n\ts_nop 1" :: "v"(ov), "v"(v), "s"(out_rows(0, mi) + (half) * 8 * NS) : "memory")
  auto read_oinv = [&]() { oinv = uni(*(volatile LAS unsigned*)(smem + LDS_CTL) != 0 ? __builtin_nanf("") : sinv); };
#define SB __builtin_amdgcn_sched_barrier(0)
#define MFMA(Z, c, w, x) do { SB;                                                           \
    if (Z) asm volatile("v_mfma_f32_16x16x128_f8f6f4 %0, %1, %2, 0" : "=&v"(c) : "v"(w), "v"(x));   \
    else asm volatile("v_mfma_f32_16x16x128_f8f6f4 %0, %1, %2, %0" : "+v"(c) : "v"(w), "v"(x));     \
    SB; } while (0)
#define MM(Z, C, H, k) MFMA(Z, C[(k) >> 1][(k) & 1], wf[H][(k) & 1], ((k) >> 1) < MI ? xf[H][((k) >> 1) & (MI - 1)] : xe[((k) >> 1) & 1])
#define DQ(D, q) do { c0[D][q] = silu(c0[D][q]); } while (0)
#define DV(M, L, k) do { if ((M) >= 0) {                                                                                 \
    if ((k) == ((L) ? 4 : 0)) DQ((M) < 0 ? 0 : (M), 0);                                                                  \
    if ((k) == ((L) ? 5 : 2)) DQ((M) < 0 ? 0 : (M), 1);                                                                  \
    if ((k) == ((L) ? 6 : 3)) { split(c0[(M) < 0 ? 0 : (M)]); SB; t0 = turn(c0[(M) < 0 ? 0 : (M)][0]); t1 = turn(c0[(M) < 0 ? 0 : (M)][1]); }   \
    if ((k) == 7) { SB; PUT(t0, M, 0); PUT(t1, M, 1); }                                                                  \
  } } while (0)
#define STEP(Z, C, H, X, M, L) do {                                                         \
    barrier();                                                                              \
    if (L) read_oinv();                                                                     \
    if ((X) > 0) { MM(Z, C, H, 0); read_e(0); }                                             \
    if ((X) > 1) { MM(Z, C, H, 1); read_e(1); }                                             \
    MM(Z, C, H, (X) + 0); read_w(wf[(H) ^ 1][0], 0); DV(M, L, 0);                           \
    MM(Z, C, H, (X) + 1); read_w(wf[(H) ^ 1][1], 1); DV(M, L, 1);                           \
    MM(Z, C, H, (X) + 2); read_x(xf[(H) ^ 1][0], 0); DV(M, L, 2);                           \
    MM(Z, C, H, (X) + 3); read_x(xf[(H) ^ 1][1], 1); DV(M, L, 3);                           \
    MM(Z, C, H, (X) + 4); read_x(xf[(H) ^ 1][2], 2); DV(M, L, 4);                           \
    MM(Z, C, H, (X) + 5); read_x(xf[(H) ^ 1][3], 3); DV(M, L, 5);                           \
    MM(Z, C, H, (X) + 6); DV(M, L, 6);                                                      \
    MM(Z, C, H, (X) + 7); DV(M, L, 7);                                                      \
    if ((X) == 1) MM(Z, C, H, 9);                                                           \
    if ((X) == 2) { MM(Z, C, H, 10); MM(Z, C, H, 11); }                                     \
    next_read_slot();                                                                       \
    SB;                                                                                     \
    __builtin_amdgcn_s_waitcnt(0xC07F);                                                     \
  } while (0)

  warm(a, lds0, idx, xcd, wave, lane);
  {
    float sx, sw;
    asm volatile("s_load_dword %0, %2, 0x0\n\ts_load_dword %1, %3, 0x0\n\ts_waitcnt lgkmcnt(0)" : "=&s"(sx), "=&s"(sw) : "s"(a.sx), "s"(a.sw) : "memory");
    sinv = uni(1.f / (sx * sw));
    nl2e = uni(-1.44269504088896341f * (sx * sw));
    asm volatile("s_mov_b32 %0, %1" : "=s"(lim) : "i"(__builtin_bit_cast(int, 60.f * 1.44269504088896341f)));
  }
  if (wm) for (int i = 0; i < DL; ++i) barrier();
  barrier();
  asm volatile("" : "+s"(sa_r), "+s"(sb_r), "+s"(se_r));
#pragma unroll
  for (int ni = 0; ni < NI; ++ni) read_w(wf[0][ni], ni);
#pragma unroll
  for (int mi = 0; mi < MI; ++mi) read_x(xf[0][mi], mi);
  next_read_slot();
  __builtin_amdgcn_s_waitcnt(0xC07F);

  STEP(1, c0, 0, NX, -1, 0);
  STEP(0, c0, 1, NX, -1, 0);
  for (int kt = 2; kt < KT - 2; kt += 2) {
    STEP(0, c0, 0, NX, -1, 0);
    STEP(0, c0, 1, NX, -1, 0);
  }
  STEP(0, c0, 0, NX, -1, 0);
  STEP(0, c0, 1, NX, 0, 1);
  STEP(1, c1, 0, 0, 1, 0);
  STEP(0, c1, 1, 0, 2, 0);
  STEP(0, c1, 0, 0, 3, 0);
  STEP(0, c1, 1, 0, 4, 0);
  if (NX == 2) {
    STEP(0, c1, 0, 0, 5, 0);
    STEP(0, c1, 1, 0, -1, 0);
  }
  for (int kt = 2 + 2 * NX; kt < KT - 2; kt += 2) {
    STEP(0, c1, 0, 0, -1, 0);
    STEP(0, c1, 1, 0, -1, 0);
  }
  MM(0, c1, 0, 0); MM(0, c1, 0, 1); MM(0, c1, 0, 2); MM(0, c1, 0, 3);
  MM(0, c1, 0, 4); MM(0, c1, 0, 5); MM(0, c1, 0, 6); MM(0, c1, 0, 7);
  const int fw = LDS_A + wm * 8192 + wn * 2048 + l16 * 128 + ((g ^ (l16 & 7)) << 4);
  const int fr = LDS_A + wm * 8192 + wn * 2048 + (lane >> 3) * 128 + (((lane & 7) ^ (lane >> 3)) << 4);
  auto finish = [&](int mi, int ni) { c1[mi][ni] = silu(c1[mi][ni]); };
  auto flip = [&](int mi) {
    const int slot = (mi % 3) * SLOT;
    *(volatile LAS v4f*)((LAS char*)smem + slot + fw) = c1[mi][0];
    *(volatile LAS v4f*)((LAS char*)smem + slot + (fw ^ 64)) = c1[mi][1];
    c1[mi][0] = *(volatile LAS v4f*)((LAS char*)smem + slot + fr);
    c1[mi][1] = *(volatile LAS v4f*)((LAS char*)smem + slot + fr + 1024);
  };
  auto leave = [&](int mi) {
    asm volatile("global_store_dwordx4 %0, %1, %2 sc0 sc1\n\ts_nop 1" :: "v"(ov), "v"(c1[mi][0]), "s"(out_rows(1, mi)) : "memory");
    asm volatile("global_store_dwordx4 %0, %1, %2 sc0 sc1\n\ts_nop 1" :: "v"(ov), "v"(c1[mi][1]), "s"(out_rows(1, mi) + 8 * NS) : "memory");
  };
  barrier();
  read_w(wf[1][0], 0); read_x(xf[1][0], 0); SB; read_w(wf[1][1], 1); SB;
  read_x(xf[1][1], 1); SB; read_x(xf[1][2], 2); SB; read_x(xf[1][3], 3); SB;
  const unsigned dead = *(volatile LAS unsigned*)(smem + LDS_CTL);
  MM(0, c1, 1, 0); MM(0, c1, 1, 1); MM(0, c1, 1, 2); MM(0, c1, 1, 3); MM(0, c1, 1, 4);
  oinv = uni(dead != 0 ? __builtin_nanf("") : sinv);
  asm volatile("s_nop 8" : "+v"(c1[0][0]), "+v"(c1[0][1]));
  finish(0, 0);
  MM(0, c1, 1, 5);
  finish(0, 1);
  MM(0, c1, 1, 6);
  flip(0);
  MM(0, c1, 1, 7);
  finish(1, 0); finish(1, 1);
  SB; flip(1); leave(0); SB;
  finish(2, 0); finish(2, 1);
  SB; flip(2); leave(1); SB;
  finish(3, 0); finish(3, 1);
  SB; flip(3); leave(2); leave(3);
}

__global__ __launch_bounds__(THREADS, 3) void gemm_ag_kernel(Args a) {
  extern __shared__ __attribute__((aligned(16))) char smem[];
  const int lane = threadIdx.x & 63, wave = __builtin_amdgcn_readfirstlane(threadIdx.x >> 6), b = blockIdx.x;
  const unsigned lds0 = (unsigned)(size_t)(LAS char*)smem;
  char* const my = a.my;
  const int xcd = b & 7, idx = b >> 3;

  if (b < 8 && threadIdx.x == 0) { unsigned xcc; asm volatile("s_getreg_b32 %0, hwreg(HW_REG_XCC_ID)" : "=s"(xcc)); __hip_atomic_store((unsigned*)(my + 8192) + b, 100u + (xcc & 15u), __ATOMIC_RELAXED, __HIP_MEMORY_SCOPE_SYSTEM); }
  if (idx < 4) {
    if (wave > 1) return;
    const unsigned die = (xcd + 7) & 7, pi = ((die >> 2) ? 0 : 4) + ((idx + die) & 3), q0 = die & 3;
    if (pi < NR - 1) {
      if (wave == 0) {
        const unsigned fg = a.first[q0];
        asm volatile("" :: "s"(fg), "s"(a.peer[pi]), "s"(a.x), "s"(a.rank), "s"(my));
        char* const peer = a.peer[pi];
        push_stream(a.x, peer + OFF_X + (size_t)a.rank * (MS * KDIM), peer + OFF_FLAGS + a.rank * FLAG_SRC, my + OFF_CNT,
                    fg & 255u, fg >> 8 & 255u, lane);
      } else {
        const unsigned ord = a.order[lane], fg = a.first[q0];
        ((LAS unsigned*)smem)[lane] = lane < 2 ? (lane + 1) << 16 | (fg >> (16 + 8 * lane) & 255u) : 0u;
        asm volatile("s_waitcnt lgkmcnt(0)\n\ts_barrier" ::: "memory");
        const unsigned call = read_call(my);
        ((LAS unsigned*)(smem + 256))[lane] = ord;
        char* const tk = my + OFF_TICKET + pi * 512;
        if (q0 == 0 && lane == 0)
          __hip_atomic_store((unsigned*)(tk + ((call + 2) & 3u) * 128), 0u, __ATOMIC_RELAXED, __HIP_MEMORY_SCOPE_AGENT);
        deal_units(tk + (call & 3u) * 128, 2 * (DD / GS) * PW);
      }
      barrier();
    }
    if (wave == 0 && lane == 0) __hip_atomic_fetch_add((unsigned*)(my + OFF_CNT), 1u, __ATOMIC_RELAXED, __HIP_MEMORY_SCOPE_AGENT);
    return;
  }

  const int si = (idx - 4) >> 2, rsub = ((xcd >> 2) ^ si) & 1, cblk = (xcd & 3) * 4 + (idx & 3);
  const int src = (a.rank + 1 + si) & 7;
  const bool two = si < 2 && rsub == 0;
  const int dl = two ? SKEW - 1 : SKEW;
  if (wave == 8 || wave == 11) {
    const int h = wave == 11;
    unsigned* const tmo = (unsigned*)(my + OFF_TMO);
    const unsigned call = read_call(my);
    if (h == 0 && lane == 0) *(LAS unsigned*)(smem + LDS_CTL) = 0;
    const char* const fl = my + OFF_FLAGS + src * FLAG_SRC + rsub * 1024 + h * 256;
    const unsigned char* p = (const unsigned char*)my + OFF_X + (size_t)(call & 1u) * X_SLOT + (size_t)src * (MS * KDIM) + rsub * (BM * 128) + h * 8192;
    const unsigned v0 = (lane >> 3) * 128 + (((lane & 7) ^ (lane >> 4)) << 4), v1 = v0 ^ 64u;
    const unsigned img = lds0 + LDS_POLL + h * 256;
    asm volatile("ds_write_b32 %0, %1\n\ts_waitcnt lgkmcnt(0)" :: "v"(img + lane * 4), "v"(0u) : "memory");
    int ready = 0, issued = 0;
    auto poll = [&]() {
      asm volatile("s_mov_b32 m0, %2\n\ts_nop 0\n\tglobal_load_lds_dword %0, %1 sc0 sc1" :: "v"(lane * 4), "s"(fl), "s"(img) : "memory");
    };
    auto count = [&](unsigned v) {
      const unsigned long long m = ~(__ballot(v >= call || lane < ready));
      ready = __builtin_amdgcn_readfirstlane(m ? __builtin_ctzll(m) : 2 * KT);
    };
    auto issue = [&]() {
      dma_half(p, v0, v1, lds0 + LDS_A + (issued & (NA - 1)) * SLOT + h * 8192);
      p += SROWS * 128;
      ++issued;
    };
    auto check_tmo = [&]() {
      unsigned tv;
      asm volatile("s_waitcnt vmcnt(0)\n\tds_read_b32 %0, %1\n\ts_waitcnt lgkmcnt(0)" : "=v"(tv) : "v"(lds0 + LDS_TMO + h * 256) : "memory");
      if ((unsigned)__builtin_amdgcn_readfirstlane(tv) == call) {
        *(LAS unsigned*)(smem + LDS_CTL) = 1;
        asm volatile("s_waitcnt lgkmcnt(0)" ::: "memory");
      }
    };
    auto round = [&](int j, int mode) {
      const int lim = j < 1 ? NA : min(j + NA - 1, 2 * KT);
      if (mode == 2) check_tmo();
      while (issued < min(ready, lim)) issue();
      if (issued <= j) {
        const unsigned long long t0 = __builtin_amdgcn_s_memrealtime();
        if (j > 0) { poll(); __builtin_amdgcn_s_sleep(3); }
        for (unsigned polls = 1;; ++polls) {
          unsigned v;
          poll();
          if (j > 0) asm volatile("s_waitcnt vmcnt(1)\n\tds_read_b32 %0, %1\n\ts_waitcnt lgkmcnt(0)" : "=v"(v) : "v"(img + lane * 4) : "memory");
          else asm volatile("s_waitcnt vmcnt(0)\n\tds_read_b32 %0, %1\n\ts_waitcnt lgkmcnt(0)" : "=v"(v) : "v"(img + lane * 4) : "memory");
          count(v);
          if (ready > j) break;
          if ((polls & 31u) == 0 &&
              __builtin_amdgcn_readfirstlane(__hip_atomic_load(tmo, __ATOMIC_RELAXED, __HIP_MEMORY_SCOPE_AGENT) == call
                                             || __builtin_amdgcn_s_memrealtime() - t0 > DEADLINE_TICKS)) {
            __hip_atomic_store(tmo, call, __ATOMIC_RELAXED, __HIP_MEMORY_SCOPE_AGENT);
            *(LAS unsigned*)(smem + LDS_CTL) = 1;
            asm volatile("s_waitcnt lgkmcnt(0)" ::: "memory");
            ready = 2 * KT;
            break;
          }
        }
        while (issued < min(ready, lim)) issue();
      }
      if (mode & 1)
        asm volatile("s_mov_b32 m0, %2\n\ts_nop 0\n\tglobal_load_lds_dword %0, %1 sc1" :: "v"(0u), "s"(tmo), "s"(lds0 + LDS_TMO + h * 256) : "memory");
      if (mode == 3) check_tmo();
      const int ahead = issued - j - 1;
      if (ready <= lim + 1 && ready < 2 * KT) {
        unsigned v;
        poll();
#define CLOSE(n) asm volatile("s_waitcnt vmcnt(" #n ")\n\tds_read_b32 %0, %1\n\ts_barrier\n\ts_waitcnt lgkmcnt(0)" : "=v"(v) : "v"(img + lane * 4) : "memory")
        if (ahead >= 2) CLOSE(17); else if (ahead == 1) CLOSE(9); else CLOSE(1);
        count(v);
      } else {
        if (ahead >= 2) wait_vm<16>(); else if (ahead == 1) wait_vm<8>(); else wait_vm<0>();
        barrier();
      }
    };
    __builtin_amdgcn_s_setprio(2);
    int j = h ? -dl : 0;
#pragma unroll 1
    for (; j < KT - 1; ++j) round(j, 0);
    round(KT - 1, 1);
    round(KT, 2);
#pragma unroll 1
    for (j = KT + 1; j < 2 * KT - 1; ++j) round(j, 0);
    round(2 * KT - 1, 3);
    if (h == 0 && lane == 0) __hip_atomic_fetch_add((unsigned*)(my + OFF_CNT), 1u, __ATOMIC_RELAXED, __HIP_MEMORY_SCOPE_AGENT);
    return;
  }
  if (wave >= 9) {
    const int h = wave - 9, nbx = two ? NB - 1 : NB;
    const bool local = h == 0 || two;
    __builtin_amdgcn_s_setprio(2);
    unsigned wv[8];
#pragma unroll
    for (int i = 0; i < 8; ++i) wv[i] = (8 * i + (lane >> 3)) * NGLOB + ((((lane & 7) ^ (lane >> 4)) << 4) ^ ((i & 2) << 5));
    const unsigned e0 = (lane >> 3) * KDIM + (((lane & 7) ^ (lane >> 4)) << 4), e1 = (e0 ^ 64u) + 8 * KDIM;
    const unsigned char* const wb = a.w + (size_t)(64 * h) * NGLOB + a.rank * NS + cblk * BN;
    const unsigned char* const ep = a.x + (size_t)(h == 0 ? (2 * si + rsub) * 32 : 448 + (si & 1) * 32) * KDIM;
    const unsigned ebase = lds0 + LDS_B + nbx * SLOT + h * ESLOT, estep = two ? 2 * ESLOT : ESLOT;
    int ib = 0, sb = 0;
    unsigned seq = 0, doneb = 0, donee = 0;
    auto issue_b = [&]() {
      const unsigned char* p = wb + (size_t)(ib & (KT - 1)) * (128 * NGLOB);
#define DMAW(r) "global_load_lds_dwordx4 " r ", %8\n\ts_add_u32 m0, m0, 0x400\n\ts_nop 0\n\t"
      asm volatile("s_mov_b32 m0, %9\n\ts_nop 0\n\t" DMAW("%0") DMAW("%1") DMAW("%2") DMAW("%3") DMAW("%4") DMAW("%5") DMAW("%6")
                   "global_load_lds_dwordx4 %7, %8"
                   :: "v"(wv[0]), "v"(wv[1]), "v"(wv[2]), "v"(wv[3]), "v"(wv[4]), "v"(wv[5]), "v"(wv[6]), "v"(wv[7]), "s"(p),
                      "s"(lds0 + LDS_B + sb * SLOT + h * 8192) : "memory", "scc");
      seq += 8;
      doneb = lane == ib ? seq : doneb;
      ++ib; sb = (sb == nbx - 1) ? 0 : sb + 1;
    };
    auto issue_e = [&](int k, int m) {
      asm volatile("s_mov_b32 m0, %3\n\ts_nop 0\n\tglobal_load_lds_dwordx4 %0, %2\n\ts_add_u32 m0, m0, 0x400\n\ts_nop 0\n\tglobal_load_lds_dwordx4 %1, %2"
                   :: "v"(e0), "v"(e1), "s"(ep + (size_t)(m * 16) * KDIM + k * 128), "s"(ebase + (k % NE) * estep + m * 2048) : "memory", "scc");
      seq += 2;
      donee = lane == m * 32 + k ? seq : donee;
    };
    issue_b(); issue_b();
    if (local) { issue_e(0, 0); issue_e(1, 0); issue_e(0, 1); issue_e(1, 1); }
    while (ib < nbx) issue_b();
    constexpr int DIST = NB - 2 - SKEW;
    static_assert(DIST == 2, "counted waits below");
    auto round = [&](int G) {
      unsigned need = 0;
      if (local) {
        if (G >= 2 && G < KT) issue_e(G, 0);
        if (G - dl >= 2 && G - dl < KT) issue_e(G - dl, 1);
        if (G >= 1 && G - 1 < KT) need = __builtin_amdgcn_readlane(donee, G - 1);
        if (G - 1 - dl >= 0 && G - 1 - dl < KT) need = max(need, (unsigned)__builtin_amdgcn_readlane(donee, 32 + G - 1 - dl));
      }
      if (G - 2 - dl >= 0 && ib < 2 * KT) issue_b();
      wait_vm_n(seq - max(need, (unsigned)__builtin_amdgcn_readlane(doneb, G)));
      barrier();
    };
    int G = 0;
#pragma unroll 1
    for (; G < 6; ++G) round(G);
    if (local) {
#pragma unroll 1
      for (; G < KT; ++G) {
        issue_e(G, 0); issue_e(G - dl, 1); issue_b();
        wait_vm<20>();
        barrier();
      }
#pragma unroll 1
      for (; G < KT + 6; ++G) round(G);
    }
    for (; ib < 2 * KT; ++G) {
      issue_b();
      wait_vm<8 * DIST>();
      barrier();
    }
#pragma unroll 1
    for (; G < 2 * KT; ++G) round(G);
    return;
  }
  if (two) compute<2>(a, smem, wave, lane, si, rsub, cblk, src, idx, xcd);
  else compute<1>(a, smem, wave, lane, si, rsub, cblk, src, idx, xcd);
}

}

extern "C" void kernel_launch(int rank, int nranks, ncclComm_t comm, hipStream_t stream,
                              void* const* d_in, const int* in_sizes, int n_in,
                              void* d_out, int out_size, const int* in_dtypes, int out_dtype,
                              void* const* d_out_all, void* const* d_ws_all, size_t ws_bytes) {
  if (nranks != NR || (unsigned)rank >= (unsigned)NR || n_in != 4 || in_sizes[0] != MS * KDIM || in_sizes[1] != KDIM * NGLOB ||
      out_size != NR * MS * NS || ws_bytes < WS_NEED) {
    fprintf(stderr, "unexpected problem geometry\n");
    return;
  }
  {
    static int calls = 0;
    int dev = -1, ndev = -1; char bus[64] = {0};
    (void)hipGetDevice(&dev); (void)hipGetDeviceCount(&ndev); (void)hipDeviceGetPCIBusId(bus, 64, dev);
    fprintf(stderr, "DIAG rank %d call %d dev %d of %d bus %s\n", rank, calls, dev, ndev, bus);
    if (calls++ == 1) { volatile unsigned* t = (volatile unsigned*)((char*)d_ws_all[rank] + 8192); fprintf(stderr, "DIAG rank %d xcc %u %u %u %u %u %u %u %u\n", rank, t[0], t[1], t[2], t[3], t[4], t[5], t[6], t[7]); }
  }
  HIP_CHECK(hipFuncSetAttribute((const void*)gemm_ag_kernel, hipFuncAttributeMaxDynamicSharedMemorySize, LDS_BYTES));
  Args a;
  a.x = (const unsigned char*)d_in[0]; a.w = (const unsigned char*)d_in[1];
  a.sx = (const float*)d_in[2]; a.sw = (const float*)d_in[3]; a.out = (float*)d_out;
  a.my = (char*)d_ws_all[rank];
  int n = 0;
  for (int same = 0; same < 2; ++same)
    for (int i = 1; i < NR; ++i) {
      const int p = (rank + i) % NR;
      if (((0xB2 >> p & 1) == (0xB2 >> rank & 1)) == (same == 1)) a.peer[n++] = (char*)d_ws_all[p];
    }
  unsigned char order[256];
  int pos = 0;
  for (int t = 0; pos < 256; ++t)
    for (int c = 0; c < 4; ++c) {
      const int h = c >> 1, r = c & 1, k = t - SKEW * h - LEAD * r;
      if (k >= 0 && k < 2 * KT) order[pos++] = (unsigned char)(4 * k + 2 * r + h);
    }
  for (int i = 0; i < 64; ++i) a.order[i] = order[4 * i] | order[4 * i + 1] << 8 | order[4 * i + 2] << 16 | (unsigned)order[4 * i + 3] << 24;
  for (int q = 0; q < PW; ++q) a.first[q] = order[q] | order[q + 4] << 8 | order[q + 8] << 16 | (unsigned)order[q + 12] << 24;
  a.rank = rank; a.pad = 0;
  gemm_ag_kernel<<<GRID, THREADS, LDS_BYTES, stream>>>(a);
  {
    static int c2 = 0;
    if (c2++ == 0) {
      volatile unsigned* t = (volatile unsigned*)((char*)d_ws_all[rank] + 8192);
      for (long i = 0; i < 300000 && (t[0] == 0 || t[7] == 0); ++i) { }
      fprintf(stderr, "DIAG2 rank %d xcc %u %u %u %u %u %u %u %u\n", rank, t[0], t[1], t[2], t[3], t[4], t[5], t[6], t[7]);
    }
  }
  HIP_CHECK(hipGetLastError());
}
```
